```python
import numpy as np

kernel_name = "gpu_gemm_ar_m4096_k4096_n16384_w8a8_dqsilu"


ABSMAX_RTOL = 0.001

COMPUTE_DTYPE = "int8"
EPILOGUE = "dequant_silu"
IN_QUANT = "int8"
OUT_QUANT = None
IN_DTYPES = {"x": "int8", "w_mat": "int8",
             "scale_x": "float32", "scale_w": "float32"}
WORLD_SIZE = 8

def setup_inputs():

    x = np.random.randint(-127, 128, size=(4096, 4096)).astype(np.int8)
    w_mat = np.random.randint(-127, 128, size=(4096, 16384)).astype(np.int8)
    scale_x = (0.02 * (1.0 + np.random.rand(1))).astype(np.float32)
    scale_w = (0.02 * (1.0 + np.random.rand(1))).astype(np.float32)
    return {"x": x, "w_mat": w_mat, "scale_x": scale_x, "scale_w": scale_w}

def reference(x, w_mat, scale_x, scale_w):
    acc = x.astype(np.float64) @ w_mat.astype(np.float64)
    y = (acc.astype(np.float32) * (scale_x * scale_w)).astype(np.float32)
    return (y / (1.0 + np.exp(-np.clip(y, -60.0, 60.0)))).astype(np.float32)
```

```cpp
#include <hip/hip_runtime.h>
#include <rccl/rccl.h>
#include <cstdio>
#include <cstdint>

#define HIP_CHECK(expr) do { hipError_t _e = (expr); if (_e != hipSuccess) { \
    fprintf(stderr, "HIP error %s at %s:%d\n", hipGetErrorString(_e), __FILE__, __LINE__); \
    return; } } while (0)

namespace wk {

#define LAS __attribute__((address_space(3)))
typedef int i32x4 __attribute__((ext_vector_type(4)));
typedef float f32x4 __attribute__((ext_vector_type(4)));
typedef unsigned u32x4 __attribute__((ext_vector_type(4)));

constexpr int GM = 4096, GN = 16384, KSH = 512, NR = 8;
constexpr unsigned PANEL = 16384;
constexpr size_t A_SRC = 2u << 20, B_SRC = 8u << 20;
constexpr size_t KT_STR = 512u << 10, KH_STR = 256u << 10, R_STR = 2u << 20;
constexpr size_t A_BYTES = NR * A_SRC, B_BYTES = NR * B_SRC, BUF_BYTES = A_BYTES + B_BYTES;
constexpr size_t DATA_OFF = 1u << 20;
constexpr size_t WS_NEED = DATA_OFF + 2 * BUF_BYTES;
constexpr size_t OFF_TICKET0 = 0, OFF_TICKET1 = 128, OFF_TMO = 256, OFF_CNT = 1024;

#define WK_CW 2
constexpr int CW = WK_CW, NCOMM = 8 * CW, GW = 32 - CW;
constexpr int NUNITS = 20;
constexpr unsigned LDS_STAGE = 4 * 2 * PANEL, LDS_BYTES = LDS_STAGE + 64;
constexpr unsigned long long DEADLINE = 40000000ull;
constexpr int GRID = 256, BLOCK = 512;

struct Params {
    const int8_t* x; const int8_t* w; const float* sx; const float* sw;
    float* out; char* ws; void* const* ws_all; int rank; int nranks;
};

#define WAIT_V(n) asm volatile("s_waitcnt vmcnt(" #n ")" ::: "memory")
#define WAIT_L(n) asm volatile("s_waitcnt lgkmcnt(" #n ")" ::: "memory")
#define BAR __builtin_amdgcn_s_barrier()
#define SCHED __builtin_amdgcn_sched_barrier(0)

__device__ __forceinline__ unsigned call_number(char* ws, size_t off, LAS unsigned* slot) {
    if (threadIdx.x == 0) {
        unsigned t = __hip_atomic_fetch_add((unsigned*)(ws + off), 1u, __ATOMIC_RELAXED, __HIP_MEMORY_SCOPE_AGENT);
        *slot = t / gridDim.x + 1u;
    }
    __syncthreads();
    unsigned c = *slot;
    __syncthreads();
    return __builtin_amdgcn_readfirstlane(c);
}

__device__ __forceinline__ float silu_f(float y) {
    const float t = __builtin_fminf(__builtin_fmaxf(y, -60.0f), 60.0f);
    const float e = __builtin_amdgcn_exp2f(t * -1.4426950408889634f);
    return y * __builtin_amdgcn_rcpf(1.0f + e);
}

__device__ __forceinline__ void step_of(int s, int rank, int& src, int& kq) {
    const int j = s - 8, q = (j * 37) >> 8;
    kq = s < 8 ? s : q;
    src = s < 8 ? rank : ((rank + 1 + j - q * 7) & 7);
}
__device__ __forceinline__ int unit_of(int s, int r) {
    const int q = ((s - 8) * 37) >> 8;
    return s < 8 ? -1 : (r == 0 ? q : 4 + 4 * r + (q >> 1));
}

#define STAMP(c, idx) do { } while (0)
#define STAMPC(c, idx) do { } while (0)
struct Ctx {
    char* ws; const char* gA; const char* gB; LAS unsigned char* lds;
    int rank, nranks; unsigned call; int ready; bool bad;
};

__device__ __forceinline__ void wait_units(Ctx& c, int need, int wid, int lane) {
    LAS unsigned* word = (LAS unsigned*)(c.lds + LDS_STAGE);
    if (wid == 0) {
        const unsigned* cnt = (const unsigned*)(c.ws + OFF_CNT);
        unsigned* tmo = (unsigned*)(c.ws + OFF_TMO);
        const unsigned target = c.call * (unsigned)NCOMM;
        const unsigned long long t0 = __builtin_amdgcn_s_memrealtime();
        unsigned res;
        for (;;) {
            const unsigned v0 = __hip_atomic_load(cnt + lane, __ATOMIC_RELAXED, __HIP_MEMORY_SCOPE_SYSTEM);
            const unsigned v1 = __hip_atomic_load(cnt + 64 + lane, __ATOMIC_RELAXED, __HIP_MEMORY_SCOPE_SYSTEM);
            const unsigned v2 = __hip_atomic_load(cnt + 128 + lane, __ATOMIC_RELAXED, __HIP_MEMORY_SCOPE_SYSTEM);
            const bool self = (lane & 7) == c.rank;
            const unsigned long long b0 = __ballot(self || v0 >= target), b1 = __ballot(self || v1 >= target), b2 = __ballot(self || v2 >= target);
            const unsigned long long n0 = ~b0, n1 = ~b1, n2 = ~b2;
            int done;
            if (n0) done = __builtin_ctzll(n0) >> 3; else if (n1) done = 8 + (__builtin_ctzll(n1) >> 3); else if (n2) done = 16 + (__builtin_ctzll(n2) >> 3); else done = 24;
            if (done > need) { res = (unsigned)done; break; }
            if (__hip_atomic_load(tmo, __ATOMIC_RELAXED, __HIP_MEMORY_SCOPE_AGENT) == c.call ||
                __builtin_amdgcn_s_memrealtime() - t0 > DEADLINE) {
                __hip_atomic_store(tmo, c.call, __ATOMIC_RELAXED, __HIP_MEMORY_SCOPE_AGENT);
                res = 24u | 0x100u; break;
            }
            __builtin_amdgcn_s_sleep(4);
        }
        __builtin_amdgcn_fence(__ATOMIC_ACQUIRE, "agent");
        WAIT_V(0);
        if (lane == 0) *word = res;
        WAIT_L(0);
    }
    asm volatile("" ::: "memory"); BAR; asm volatile("" ::: "memory");
    const unsigned r = __builtin_amdgcn_readfirstlane(*word);
    WAIT_L(0);
    asm volatile("" ::: "memory"); BAR; asm volatile("" ::: "memory");
    c.ready = (int)(r & 0xffu) - 1;
    c.bad = c.bad || (r & 0x100u);
}

__device__ __forceinline__ void tile_of(int xg, int e, int& pm, int& pn) {
    const int r = e >> 5, idx = e & 31;
    pm = 4 * (xg >> 1) + (idx & 3);
    pn = 16 * r + 8 * (xg & 1) + (idx >> 2);
}

template <int FM, int KS>
__device__ __forceinline__ void gemm_stream(Ctx& c, const Params& p, float scale, int xg, int e0, int estep, int ntiles, int sub) {
    constexpr int FN = FM / 2;
    const int tid = threadIdx.x, wid = __builtin_amdgcn_readfirstlane(tid >> 6), lane = tid & 63;
    const int wr = wid >> 2, wc = wid & 3, fr = lane & 15, fq = lane >> 4;
    constexpr int PW = FM / 4;
    constexpr unsigned SUB = 2 * FM * 1024 * 2;
    constexpr unsigned SLOT = KS * SUB;
    constexpr unsigned BOFF = 2 * FM * 1024;
    constexpr int NG = (FM + FN) * KS, NM = FM * FN * KS, NL = 2 * PW * KS;
    constexpr int NST = 64 / KS;
    static_assert(4 * SLOT <= LDS_STAGE, "ring");
    const int rank = c.rank;
    const int subA = (FM == 8) ? 0 : (sub >> 1) * 8, subB = (FM == 8) ? 0 : (sub & 1) * 8;
    const unsigned gvoff = (unsigned)(wid * PW) * 1024u + (unsigned)lane * 16u;
    const unsigned ldsw = (unsigned)(wid * PW) * 1024u;
    const unsigned aoff = (unsigned)(wr * FM) * 1024u + (unsigned)lane * 16u, boff = BOFF + (unsigned)(wc * FN) * 1024u + (unsigned)lane * 16u;

    i32x4 acc[FM][FN];
    i32x4 fa[2][KS][FM], fb[2][KS][FN];

    int pm, pn; tile_of(xg, e0, pm, pn);
    const char* cA = c.gA + (size_t)pm * PANEL + subA * 1024;
    const char* cB = c.gB + (size_t)(pn >> 4) * R_STR + (size_t)(pn & 15) * PANEL + subB * 1024;
    int rr = pn >> 4;

    const unsigned ldsbase = (unsigned)(uintptr_t)c.lds;
    const unsigned raLo = ldsbase + aoff, rbLo = ldsbase + boff, raHi = raLo + 2 * SLOT, rbHi = rbLo + 2 * SLOT;
#define MFMA_I8(ACC, B, A) asm volatile("v_mfma_i32_16x16x64_i8 %0, %1, %2, %0" : "+v"(ACC) : "v"(B), "v"(A))
#define DSREAD(DST, ADDR, OFF) asm volatile("ds_read_b128 %0, %1 offset:%2" : "=v"(DST) : "v"(ADDR), "i"(OFF))
#define RDA(set, ks, i, slot) DSREAD(fa[set][ks][i], ((slot) & 2) ? raHi : raLo, ((slot) & 1) * SLOT + (ks) * SUB + (i) * 1024)
#define RDB(set, ks, i, slot) DSREAD(fb[set][ks][i], ((slot) & 2) ? rbHi : rbLo, ((slot) & 1) * SLOT + (ks) * SUB + (i) * 1024)
#define GLDS(gp, slot, ks, isB, j) __builtin_amdgcn_global_load_lds((const unsigned*)((gp) + (j) * 1024), \
        (LAS unsigned*)(c.lds + (slot) * SLOT + (ks) * SUB + (isB) * BOFF + ldsw + (j) * 1024), 16, 0, 0)

#pragma unroll
    for (int st = 0; st < 4; ++st)
#pragma unroll
        for (int ks = 0; ks < KS; ++ks) {
            const size_t so = (size_t)rank * 0 + (size_t)(st * KS + ks) * KH_STR;
            const char* a = cA + (size_t)rank * A_SRC + so + gvoff;
            const char* b = cB + (size_t)rank * B_SRC + so + gvoff;
#pragma unroll
            for (int j = 0; j < PW; ++j) GLDS(a, st, ks, 0, j);
#pragma unroll
            for (int j = 0; j < PW; ++j) GLDS(b, st, ks, 1, j);
        }
    asm volatile("s_waitcnt vmcnt(%0)" :: "i"(3 * NL) : "memory");
    BAR;
#pragma unroll
    for (int ks = 0; ks < KS; ++ks) {
#pragma unroll
        for (int i = 0; i < FN; ++i) RDB(0, ks, i, 0);
#pragma unroll
        for (int i = 0; i < FM; ++i) RDA(0, ks, i, 0);
    }

    for (int ti = 0; ti < ntiles; ++ti) {
        const bool has_next = ti + 1 < ntiles;
        int npm = pm, npn = pn;
        if (has_next) tile_of(xg, e0 + (ti + 1) * estep, npm, npn);
        const char* nA = c.gA + (size_t)npm * PANEL + subA * 1024;
        const char* nB = c.gB + (size_t)(npn >> 4) * R_STR + (size_t)(npn & 15) * PANEL + subB * 1024;
#pragma unroll
        for (int m = 0; m < FM; ++m)
#pragma unroll
            for (int n = 0; n < FN; ++n) acc[m][n] = (i32x4){0, 0, 0, 0};
        for (int s4 = 0; s4 < NST; s4 += 4) {
            const bool last = (s4 == NST - 4);
            if (!last) { const int need = unit_of((s4 + 8) * KS - 1, rr); if (need > c.ready) wait_units(c, need, wid, lane); }
            const char* tA = last ? nA : cA; const char* tB = last ? nB : cB;
            const int sn = last ? 0 : s4 + 4;
#pragma unroll
            for (int u = 0; u < 4; ++u) {
                const char* ga[KS]; const char* gb[KS];
#pragma unroll
                for (int ks = 0; ks < KS; ++ks) {
                    int src, kq; step_of((sn + u) * KS + ks, rank, src, kq);
                    const size_t so = (size_t)kq * KH_STR;
                    ga[ks] = tA + (size_t)src * A_SRC + so + gvoff;
                    gb[ks] = tB + (size_t)src * B_SRC + so + gvoff;
                }
                asm volatile("s_waitcnt vmcnt(%0)" :: "i"(2 * NL) : "memory");
                WAIT_L(0);
                BAR;
                SCHED;
#pragma unroll
                for (int g = 0; g < NG; ++g) {
#pragma unroll
                    for (int idx = g * NM / NG; idx < (g + 1) * NM / NG; ++idx) {
                        const int ks = idx / (FM * FN), m = (idx / FN) % FM, n = idx % FN;
                        MFMA_I8(acc[m][n], fb[u & 1][ks][n], fa[u & 1][ks][m]);
                    }
                    { const int ks = g / (FM + FN), i = g % (FM + FN);
                      if (i < FN) RDB((u + 1) & 1, ks, i, (u + 1) & 3); else RDA((u + 1) & 1, ks, i - FN, (u + 1) & 3); }
                    if (g % 3 == 1 && g / 3 < NL) {
                        const int l = g / 3, ks = l / (2 * PW), j = l % (2 * PW);
                        if (j < PW) GLDS(ga[ks], u, ks, 0, j); else GLDS(gb[ks], u, ks, 1, j - PW);
                    }
                }
                SCHED;
            }
        }
        asm volatile("s_nop 15\n\ts_nop 15" ::: "memory");
        if (FM == 8) { STAMP(c, 24 + ti); if (ti == 3) STAMPC(c, 29); }
        {
            float* o = p.out + (size_t)(pm * 256 + subA * 16 + wr * (FM * 16) + fr) * GN + (pn * 256 + subB * 16 + wc * (FN * 16) + 4 * fq);
            const float sc = c.bad ? __builtin_nanf("") : scale;
#pragma unroll
            for (int m = 0; m < FM; ++m)
#pragma unroll
                for (int n = 0; n < FN; ++n) {
                    const i32x4 a = acc[m][n];
                    f32x4 v;
#pragma unroll
                    for (int j = 0; j < 4; ++j) v[j] = silu_f((float)a[j] * sc);
                    *(f32x4*)(o + (size_t)(m * 16) * GN + n * 16) = v;
                }
        }
        pm = npm; pn = npn; cA = nA; cB = nB; rr = pn >> 4;
        if (FM == 8) { STAMP(c, 1 + ti); if (ti == 2) STAMPC(c, 28); }
    }
    WAIT_V(0); WAIT_L(0);
    BAR;
#undef MFMA_I8
#undef DSREAD
#undef RDA
#undef RDB
#undef GLDS
}

__device__ __forceinline__ void retile(const Params& p, char* bufA, char* bufB) {
    const int tid = threadIdx.x, bid = blockIdx.x;
    char* dA = bufA + (size_t)p.rank * A_SRC;
    char* dB = bufB + (size_t)p.rank * B_SRC;
    for (int cidx = bid * BLOCK + tid; cidx < (GM * KSH / 16); cidx += GRID * BLOCK) {
        const int kt = cidx >> 15, kh = (cidx >> 14) & 1, pmf = (cidx >> 6) & 255, ln = cidx & 63;
        const int row = pmf * 16 + (ln & 15), kb = kt * 128 + kh * 64 + (ln >> 4) * 16;
        const u32x4 v = *(const u32x4*)(p.x + (size_t)row * KSH + kb);
        *(u32x4*)(dA + (size_t)cidx * 16) = v;
    }
    if (tid < 128) {
        const int task = bid * 128 + tid;
        const int nb = task & 1023, kb = task >> 10;
        u32x4 rws[16];
#pragma unroll
        for (int i = 0; i < 16; ++i) rws[i] = *(const u32x4*)(p.w + (size_t)(kb * 16 + i) * GN + nb * 16);
        u32x4 o[16];
#pragma unroll
        for (int q = 0; q < 4; ++q)
#pragma unroll
            for (int wd = 0; wd < 4; ++wd) {
                const unsigned a = rws[4 * q][wd], b = rws[4 * q + 1][wd], cc = rws[4 * q + 2][wd], d = rws[4 * q + 3][wd];
                const unsigned t0 = __builtin_amdgcn_perm(b, a, 0x05010400u), t1 = __builtin_amdgcn_perm(b, a, 0x07030602u);
                const unsigned t2 = __builtin_amdgcn_perm(d, cc, 0x05010400u), t3 = __builtin_amdgcn_perm(d, cc, 0x07030602u);
                o[4 * wd + 0][q] = __builtin_amdgcn_perm(t2, t0, 0x05040100u);
                o[4 * wd + 1][q] = __builtin_amdgcn_perm(t2, t0, 0x07060302u);
                o[4 * wd + 2][q] = __builtin_amdgcn_perm(t3, t1, 0x05040100u);
                o[4 * wd + 3][q] = __builtin_amdgcn_perm(t3, t1, 0x07060302u);
            }
        const int k0 = kb * 16;
        const int r = nb >> 8, pnf = nb & 255;
        const int kt = k0 >> 7, kh = (k0 >> 6) & 1, kq = (k0 >> 4) & 3;
        char* dst = dB + (size_t)(r * 4 + kt) * KT_STR + (size_t)kh * KH_STR + (size_t)pnf * 1024 + kq * 256;
#pragma unroll
        for (int i = 0; i < 16; ++i) *(u32x4*)(dst + i * 16) = o[i];
    }
}

__device__ __forceinline__ void push_all(const Ctx& c, const Params& p, size_t bufoff, int cw) {
    const int tid = threadIdx.x;
    constexpr size_t SL = KH_STR / NCOMM;
    constexpr int NI = (int)(SL / (BLOCK * 16));
    char* peer[7];
#pragma unroll
    for (int i = 0; i < 7; ++i) { const unsigned pr = (unsigned)(c.rank + 1 + i) & 7u; peer[i] = (char*)p.ws_all[pr]; }
    for (int u = 0; u < NUNITS; ++u) {
        size_t off[2];
        const size_t lo = (size_t)cw * SL + (size_t)tid * 16;
        if (u < 8) {
            off[0] = bufoff + A_BYTES + (size_t)c.rank * B_SRC + (size_t)u * KH_STR + lo;
            off[1] = bufoff + (size_t)c.rank * A_SRC + (size_t)u * KH_STR + lo;
        } else {
            off[0] = bufoff + A_BYTES + (size_t)c.rank * B_SRC + (size_t)(u - 4) * KT_STR + lo;
            off[1] = off[0] + KH_STR;
        }
        u32x4 v[2][NI];
#pragma unroll
        for (int pc = 0; pc < 2; ++pc)
#pragma unroll
            for (int i = 0; i < NI; ++i) v[pc][i] = *(const u32x4*)(c.ws + off[pc] + (size_t)i * (BLOCK * 16));
#pragma unroll
        for (int pc = 0; pc < 2; ++pc)
#pragma unroll
            for (int i = 0; i < NI; ++i)
#pragma unroll
                for (int q = 0; q < 7; ++q)
                    asm volatile("global_store_dwordx4 %0, %1, off sc0 sc1\n\ts_nop 1" :: "v"(peer[q] + off[pc] + (size_t)i * (BLOCK * 16)), "v"(v[pc][i]) : "memory");
        WAIT_V(0);
        __syncthreads();
        if (tid < 7) {
            const unsigned pr = (unsigned)(c.rank + 1 + tid) & 7u;
            unsigned* cnt = (unsigned*)((char*)p.ws_all[pr] + OFF_CNT) + u * 8 + c.rank;
            __hip_atomic_fetch_add(cnt, 1u, __ATOMIC_RELAXED, __HIP_MEMORY_SCOPE_SYSTEM);
        }
        if (u >= 4) STAMP(c, 4 + u);
    }
}

__global__ void __launch_bounds__(BLOCK, 2) fused_kernel(Params p, int phase) {
    extern __shared__ __attribute__((aligned(16))) unsigned char shm[];
    LAS unsigned char* lds = (LAS unsigned char*)shm;
    LAS unsigned* slot = (LAS unsigned*)(lds + LDS_STAGE + 16);
    if (phase == 0) {
        const unsigned call = call_number(p.ws, OFF_TICKET0, slot);
        char* buf = p.ws + DATA_OFF + (size_t)(call & 1u) * BUF_BYTES;
        retile(p, buf, buf + A_BYTES);
        return;
    }
    Ctx c;
    c.call = call_number(p.ws, OFF_TICKET1, slot);
    const size_t bufoff = DATA_OFF + (size_t)(c.call & 1u) * BUF_BYTES;
    c.ws = p.ws; c.gA = p.ws + bufoff; c.gB = c.gA + A_BYTES; c.lds = lds;
    c.rank = p.rank; c.nranks = p.nranks; c.ready = -1; c.bad = false;
    const float scale = p.sx[0] * p.sw[0];
    const int xg = blockIdx.x & 7, j = blockIdx.x >> 3;
    STAMP(c, 0);
    if (j < CW) push_all(c, p, bufoff, j * 8 + xg);
    else gemm_stream<8, 1>(c, p, scale, xg, j - CW, GW, 4, 0);
    for (int qi = j; qi < 16 * CW; qi += 32) {
        gemm_stream<4, 2>(c, p, scale, xg, 4 * GW + (qi >> 2), 0, 1, qi & 3);
    }
    STAMP(c, 5);
}

}

extern "C" void kernel_launch(int rank, int nranks, ncclComm_t comm,
                              hipStream_t stream,
                              void* const* d_in, const int* in_sizes, int n_in,
                              void* d_out, int out_size,
                              const int* in_dtypes, int out_dtype,
                              void* const* d_out_all,
                              void* const* d_ws_all, size_t ws_bytes) {
    (void)comm; (void)in_sizes; (void)n_in; (void)out_size; (void)in_dtypes; (void)out_dtype; (void)d_out_all;
    if (nranks != wk::NR || ws_bytes < wk::WS_NEED || (unsigned)rank >= (unsigned)nranks) {
        fprintf(stderr, "unsupported configuration (nranks %d, ws %zu)\n", nranks, ws_bytes);
        return;
    }
    static bool attr_set = false;
    if (!attr_set)
    {
        HIP_CHECK(hipFuncSetAttribute((const void*)wk::fused_kernel, hipFuncAttributeMaxDynamicSharedMemorySize, (int)wk::LDS_BYTES));
        attr_set = true;
    }
    wk::Params p;
    p.x = (const int8_t*)d_in[0]; p.w = (const int8_t*)d_in[1];
    p.sx = (const float*)d_in[2]; p.sw = (const float*)d_in[3];
    p.out = (float*)d_out; p.ws = (char*)d_ws_all[rank]; p.ws_all = d_ws_all;
    p.rank = rank; p.nranks = nranks;
    hipLaunchKernelGGL(wk::fused_kernel, dim3(wk::GRID), dim3(wk::BLOCK), wk::LDS_BYTES, stream, p, 0);
    hipLaunchKernelGGL(wk::fused_kernel, dim3(wk::GRID), dim3(wk::BLOCK), wk::LDS_BYTES, stream, p, 1);
    HIP_CHECK(hipGetLastError());
}
```

```cpp
#include <hip/hip_runtime.h>
#include <rccl/rccl.h>
#include <cstdio>
#include <cstdint>

#define HIP_CHECK(expr) do { hipError_t _e = (expr); if (_e != hipSuccess) { \
    fprintf(stderr, "HIP error %s at %s:%d\n", hipGetErrorString(_e), __FILE__, __LINE__); \
    return; } } while (0)

namespace wk {

#define LAS __attribute__((address_space(3)))
typedef int i32x4 __attribute__((ext_vector_type(4)));
typedef float f32x4 __attribute__((ext_vector_type(4)));
typedef unsigned u32x4 __attribute__((ext_vector_type(4)));

constexpr int GM = 4096, GN = 16384, KSH = 512, NR = 8;
constexpr unsigned PANEL = 16384;
constexpr size_t A_SRC = 2u << 20, B_SRC = 8u << 20;
constexpr size_t KT_STR = 512u << 10, KH_STR = 256u << 10, R_STR = 2u << 20;
constexpr size_t A_BYTES = NR * A_SRC, B_BYTES = NR * B_SRC, BUF_BYTES = A_BYTES + B_BYTES;
constexpr size_t DATA_OFF = 1u << 20;
constexpr size_t WS_NEED = DATA_OFF + 2 * BUF_BYTES;
constexpr size_t OFF_TICKET0 = 0, OFF_TICKET1 = 128, OFF_TMO = 256, OFF_CNT = 1024;

#define WK_CW 2
constexpr int CW = WK_CW, NCOMM = 8 * CW, GW = 32 - CW;
constexpr int NUNITS = 21;
constexpr unsigned LDS_STAGE = 4 * 2 * PANEL, LDS_BYTES = LDS_STAGE + 64;
constexpr unsigned long long DEADLINE = 40000000ull;
constexpr int WN = 4;
constexpr int GRID = 256, BLOCK = 128 * WN, NWAVE = 2 * WN, NPUSH = NCOMM * NWAVE;

struct Params {
    const int8_t* x; const int8_t* w; const float* sx; const float* sw;
    float* out; char* ws; void* const* ws_all; int rank; int nranks;
};

#define WAIT_V(n) asm volatile("s_waitcnt vmcnt(" #n ")" ::: "memory")
#define WAIT_L(n) asm volatile("s_waitcnt lgkmcnt(" #n ")" ::: "memory")
#define BAR __builtin_amdgcn_s_barrier()
#define SCHED __builtin_amdgcn_sched_barrier(0)

__device__ __forceinline__ unsigned call_number(char* ws, size_t off, LAS unsigned* slot) {
    if (threadIdx.x == 0) {
        unsigned t = __hip_atomic_fetch_add((unsigned*)(ws + off), 1u, __ATOMIC_RELAXED, __HIP_MEMORY_SCOPE_AGENT);
        *slot = t / gridDim.x + 1u;
    }
    __syncthreads();
    unsigned c = *slot;
    __syncthreads();
    return __builtin_amdgcn_readfirstlane(c);
}

__device__ __forceinline__ void step_of(int s, int rank, int& src, int& kq) {
    const int j = s - 8, q = (j * 37) >> 8;
    kq = s < 8 ? s : q;
    src = s < 8 ? rank : ((rank + 1 + j - q * 7) & 7);
}
__device__ __forceinline__ int unit_base(int pm, int pn) {
    return pn >= 20 ? 9 + 4 * ((pn - 20) >> 4) : ((pm >= 8 || pn >= 8) ? 4 : 0);
}
__device__ __forceinline__ int unit_of(int s, int ub) {
    const int q = ((s - 8) * 37) >> 8;
    return s < 8 ? -1 : ub + (q >> 1) + ((ub == 4 && q == 7) ? 1 : 0);
}

#define STAMP(c, idx) do { } while (0)
#define STAMPC(c, idx) do { } while (0)
struct Ctx {
    char* ws; const char* gA; const char* gB; LAS unsigned char* lds;
    int rank, nranks; unsigned call; int ready; bool bad;
};

__device__ __forceinline__ void wait_units(Ctx& c, int need, int wid, int lane) {
    LAS unsigned* word = (LAS unsigned*)(c.lds + LDS_STAGE);
    if (wid == 0) {
        const unsigned* cnt = (const unsigned*)(c.ws + OFF_CNT);
        unsigned* tmo = (unsigned*)(c.ws + OFF_TMO);
        const unsigned target = c.call * (unsigned)NCOMM;
        const unsigned long long t0 = __builtin_amdgcn_s_memrealtime();
        unsigned res;
        for (;;) {
            const unsigned v0 = __hip_atomic_load(cnt + need * 8 + lane, __ATOMIC_RELAXED, __HIP_MEMORY_SCOPE_SYSTEM);
            const unsigned long long n0 = ~__ballot((lane & 7) == c.rank || v0 >= target);
            const int done = need + (n0 ? (__builtin_ctzll(n0) >> 3) : 8);
            if (done > need) { res = (unsigned)done; break; }
            if (__hip_atomic_load(tmo, __ATOMIC_RELAXED, __HIP_MEMORY_SCOPE_AGENT) == c.call ||
                __builtin_amdgcn_s_memrealtime() - t0 > DEADLINE) {
                __hip_atomic_store(tmo, c.call, __ATOMIC_RELAXED, __HIP_MEMORY_SCOPE_AGENT);
                res = 32u | 0x100u; break;
            }
#define WK_SLEEP 16
            __builtin_amdgcn_s_sleep(WK_SLEEP);
        }
        __builtin_amdgcn_fence(__ATOMIC_ACQUIRE, "agent");
        WAIT_V(0);
        if (lane == 0) *word = res;
        WAIT_L(0);
    }
    asm volatile("" ::: "memory"); BAR; asm volatile("" ::: "memory");
    const unsigned r = __builtin_amdgcn_readfirstlane(*word);
    WAIT_L(0);
    asm volatile("" ::: "memory"); BAR; asm volatile("" ::: "memory");
    c.ready = (int)(r & 0xffu) - 1;
    c.bad = c.bad || (r & 0x100u);
}

__device__ __forceinline__ void tile_of(int xg, int e, int& pm, int& pn) {
    if (e < 0) { pm = -1 - e; pn = xg; return; }
    const int sl = xg + 8 * (e >> 2);
    const int grp = sl < 16 ? 2 + (sl & 1) : ((sl - 16) & 3);
    pn = sl < 16 ? (sl >> 1) : 8 + ((sl - 16) >> 2);
    pm = 4 * grp + (e & 3);
}

template <int FM, int KS>
__device__ __forceinline__ void gemm_stream(Ctx& c, const Params& p, float scale, int xg, int e0, int estep, int ntiles, int sub) {
    constexpr int FN = 2 * FM / WN;
    const int tid = threadIdx.x, wid = __builtin_amdgcn_readfirstlane(tid >> 6), lane = tid & 63;
    const int wr = wid / WN, wc = wid % WN, fr = lane & 15, fq = lane >> 4;
    constexpr int PW = 2 * FM / NWAVE;
    constexpr unsigned SUB = 2 * FM * 1024 * 2;
    constexpr unsigned SLOT = KS * SUB;
    constexpr unsigned BOFF = 2 * FM * 1024;
    constexpr int NG = (FM + FN) * KS, NM = FM * FN * KS, NL = 2 * PW * KS;
    constexpr int NST = 64 / KS;
#define WK_NPRE 8
    constexpr int NPRE = WK_NPRE * NM / 32;
    constexpr int GSP = NG / NL;
    static_assert(4 * SLOT <= LDS_STAGE, "ring");
    const int rank = c.rank;
    const int subA = (FM == 8) ? 0 : (sub >> 1) * 8, subB = (FM == 8) ? 0 : (sub & 1) * 8;
    const unsigned gvoff = (unsigned)(wid * PW) * 1024u + (unsigned)lane * 16u;
    const unsigned ldsw = (unsigned)(wid * PW) * 1024u;
    const unsigned aoff = (unsigned)(wr * FM) * 1024u + (unsigned)lane * 16u, boff = BOFF + (unsigned)(wc * FN) * 1024u + (unsigned)lane * 16u;

    i32x4 acc[FM][FN];
    i32x4 fa[2][KS][FM], fb[2][KS][FN];

    int pm, pn; tile_of(xg, e0, pm, pn);
    const char* cA = c.gA + (size_t)pm * PANEL + subA * 1024;
    const char* cB = c.gB + (size_t)(pn >> 4) * R_STR + (size_t)(pn & 15) * PANEL + subB * 1024;
    int ub = unit_base(pm, pn);

    const unsigned ldsbase = (unsigned)(uintptr_t)c.lds;
    const unsigned raLo = ldsbase + aoff, rbLo = ldsbase + boff, raHi = raLo + 2 * SLOT, rbHi = rbLo + 2 * SLOT;
#define MFMA_I8(ACC, B, A) asm volatile("v_mfma_i32_16x16x64_i8 %0, %1, %2, %0" : "+v"(ACC) : "v"(B), "v"(A))
#define DSREAD(DST, ADDR, OFF) asm volatile("ds_read_b128 %0, %1 offset:%2" : "=v"(DST) : "v"(ADDR), "i"(OFF))
#define RDA(set, ks, i, slot) DSREAD(fa[set][ks][i], ((slot) & 2) ? raHi : raLo, ((slot) & 1) * SLOT + (ks) * SUB + (i) * 1024)
#define RDB(set, ks, i, slot) DSREAD(fb[set][ks][i], ((slot) & 2) ? rbHi : rbLo, ((slot) & 1) * SLOT + (ks) * SUB + (i) * 1024)
#define GLDS(gp, slot, ks, isB, j) __builtin_amdgcn_global_load_lds((const unsigned*)((gp) + (j) * 1024), \
        (LAS unsigned*)(c.lds + (slot) * SLOT + (ks) * SUB + (isB) * BOFF + ldsw + (j) * 1024), 16, 0, 0)

#pragma unroll
    for (int st = 0; st < 4; ++st)
#pragma unroll
        for (int ks = 0; ks < KS; ++ks) {
            const size_t so = (size_t)(st * KS + ks) * KH_STR;
            const char* a = cA + (size_t)rank * A_SRC + so + gvoff;
            const char* b = cB + (size_t)rank * B_SRC + so + gvoff;
#pragma unroll
            for (int j = 0; j < PW; ++j) GLDS(a, st, ks, 0, j);
#pragma unroll
            for (int j = 0; j < PW; ++j) GLDS(b, st, ks, 1, j);
        }
    asm volatile("s_waitcnt vmcnt(%0)" :: "i"(3 * NL) : "memory");
    BAR;
#pragma unroll
    for (int ks = 0; ks < KS; ++ks) {
#pragma unroll
        for (int i = 0; i < FN; ++i) RDB(0, ks, i, 0);
#pragma unroll
        for (int i = 0; i < FM; ++i) RDA(0, ks, i, 0);
    }

    for (int ti = 0; ti < ntiles; ++ti) {
        const bool has_next = ti + 1 < ntiles;
        int npm = pm, npn = pn;
        if (has_next) tile_of(xg, e0 + (ti + 1) * estep, npm, npn);
        const char* nA = c.gA + (size_t)npm * PANEL + subA * 1024;
        const char* nB = c.gB + (size_t)(npn >> 4) * R_STR + (size_t)(npn & 15) * PANEL + subB * 1024;
#pragma unroll
        for (int m = 0; m < FM; ++m)
#pragma unroll
            for (int n = 0; n < FN; ++n) acc[m][n] = (i32x4){0, 0, 0, 0};
        for (int s4 = 0; s4 < NST; s4 += 4) {
            const bool last = (s4 == NST - 4);
            if (!last) { const int need = unit_of((s4 + 8) * KS - 1, ub); if (need > c.ready) wait_units(c, need, wid, lane); }
            const char* tA = last ? nA : cA; const char* tB = last ? nB : cB;
            const int sn = last ? 0 : s4 + 4;
#pragma unroll
            for (int u = 0; u < 4; ++u) {
                const char* ga[KS]; const char* gb[KS];
#pragma unroll
                for (int ks = 0; ks < KS; ++ks) {
                    int src, kq; step_of((sn + u) * KS + ks, rank, src, kq);
                    const size_t so = (size_t)kq * KH_STR;
                    ga[ks] = tA + (size_t)src * A_SRC + so + gvoff;
                    gb[ks] = tB + (size_t)src * B_SRC + so + gvoff;
                }
                WAIT_L(0);
                SCHED;
                asm volatile("s_nop 1" ::: "memory");
#pragma unroll
                for (int idx = 0; idx < NPRE; ++idx) {
                    const int ks = idx / (FM * FN), m = (idx / FN) % FM, n = idx % FN;
                    MFMA_I8(acc[m][n], fb[u & 1][ks][n], fa[u & 1][ks][m]);
                }
                asm volatile("s_waitcnt vmcnt(%0)" :: "i"(2 * NL) : "memory");
                BAR;
                SCHED;
#pragma unroll
                for (int g = 0; g < NG; ++g) {
                    { const int ks = g / (FM + FN), i = g % (FM + FN);
                      if (i < FN) RDB((u + 1) & 1, ks, i, (u + 1) & 3); else RDA((u + 1) & 1, ks, i - FN, (u + 1) & 3); }
                    if (g % GSP == 1 % GSP && g / GSP < NL) {
                        const int l = g / GSP, ks = l / (2 * PW), j = l % (2 * PW);
                        if (j < PW) GLDS(ga[ks], u, ks, 0, j); else GLDS(gb[ks], u, ks, 1, j - PW);
                    }
#pragma unroll
                    for (int idx = NPRE + g * (NM - NPRE) / NG; idx < NPRE + (g + 1) * (NM - NPRE) / NG; ++idx) {
                        const int ks = idx / (FM * FN), m = (idx / FN) % FM, n = idx % FN;
                        MFMA_I8(acc[m][n], fb[u & 1][ks][n], fa[u & 1][ks][m]);
                    }
                }
                SCHED;
            }
        }
        asm volatile("s_nop 15\n\ts_nop 15" ::: "memory");
        if (FM == 8) { STAMP(c, 24 + ti); if (ti == 3) STAMPC(c, 29); }
        {
            float* o = p.out + (size_t)(pm * 256 + subA * 16 + wr * (FM * 16) + fr) * GN + (pn * 256 + subB * 16 + wc * (FN * 16) + 4 * fq);
            const float sc = c.bad ? __builtin_nanf("") : scale;
            const float se = sc * -1.4426950408889634f;
#pragma unroll
            for (int m = 0; m < FM; ++m)
#pragma unroll
                for (int n = 0; n < FN; ++n) {
                    const i32x4 a = acc[m][n];
                    const f32x4 f = {(float)a[0], (float)a[1], (float)a[2], (float)a[3]};
                    const f32x4 y = f * sc, t = f * se;
                    f32x4 d;
#pragma unroll
                    for (int j = 0; j < 4; ++j) d[j] = __builtin_amdgcn_exp2f(__builtin_amdgcn_fmed3f(t[j], -86.5617024533378f, 86.5617024533378f));
                    d = d + 1.0f;
                    f32x4 r;
#pragma unroll
                    for (int j = 0; j < 4; ++j) r[j] = __builtin_amdgcn_rcpf(d[j]);
                    *(f32x4*)(o + (size_t)(m * 16) * GN + n * 16) = y * r;
                }
        }
        pm = npm; pn = npn; cA = nA; cB = nB; ub = unit_base(pm, pn);
        if (FM == 8) { STAMP(c, 1 + ti); if (ti == 2) STAMPC(c, 28); }
    }
    WAIT_V(0); WAIT_L(0);
    BAR;
#undef MFMA_I8
#undef DSREAD
#undef RDA
#undef RDB
#undef GLDS
}

__device__ __forceinline__ void retile(const Params& p, LAS unsigned* slot) {
    const int tid = threadIdx.x, bid = blockIdx.x;
    const int task = bid * BLOCK + tid;
    u32x4 xv;
    {
        const int kt = task >> 15, kh = (task >> 14) & 1, pmf = (task >> 6) & 255, ln = task & 63;
        const int row = pmf * 16 + (ln & 15), kb = kt * 128 + kh * 64 + (ln >> 4) * 16;
        xv = *(const u32x4*)(p.x + (size_t)row * KSH + kb);
    }
    const int nq = task & 4095, kb = task >> 12;
    unsigned rws[16];
#pragma unroll
    for (int i = 0; i < 16; ++i) rws[i] = *(const unsigned*)(p.w + (size_t)(kb * 16 + i) * GN + nq * 4);
    const unsigned call = call_number(p.ws, OFF_TICKET0, slot);
    char* buf = p.ws + DATA_OFF + (size_t)(call & 1u) * BUF_BYTES;
    char* dA = buf + (size_t)p.rank * A_SRC;
    char* dB = buf + A_BYTES + (size_t)p.rank * B_SRC;
    *(u32x4*)(dA + (size_t)task * 16) = xv;
    u32x4 o[4];
#pragma unroll
    for (int q = 0; q < 4; ++q) {
        const unsigned a = rws[4 * q], b = rws[4 * q + 1], cc = rws[4 * q + 2], d = rws[4 * q + 3];
        const unsigned t0 = __builtin_amdgcn_perm(b, a, 0x05010400u), t1 = __builtin_amdgcn_perm(b, a, 0x07030602u);
        const unsigned t2 = __builtin_amdgcn_perm(d, cc, 0x05010400u), t3 = __builtin_amdgcn_perm(d, cc, 0x07030602u);
        o[0][q] = __builtin_amdgcn_perm(t2, t0, 0x05040100u);
        o[1][q] = __builtin_amdgcn_perm(t2, t0, 0x07060302u);
        o[2][q] = __builtin_amdgcn_perm(t3, t1, 0x05040100u);
        o[3][q] = __builtin_amdgcn_perm(t3, t1, 0x07060302u);
    }
    const int k0 = kb * 16, nb = nq >> 2;
    const int r = nb >> 8, pnf = nb & 255;
    const int kt = k0 >> 7, kh = (k0 >> 6) & 1, kq = (k0 >> 4) & 3;
    char* dst = dB + (size_t)(r * 4 + kt) * KT_STR + (size_t)kh * KH_STR + (size_t)pnf * 1024 + kq * 256 + (nq & 3) * 64;
#pragma unroll
    for (int i = 0; i < 4; ++i) *(u32x4*)(dst + i * 16) = o[i];
}
constexpr int GRID0 = GM * KSH / 16 / BLOCK;
static_assert(2 * 20 * 16 <= 5 * NPUSH, "at most five cells per wave and unit");
static_assert(CW == 2, "schedule: 30 GEMM workgroups x 4 tiles per XCD group");
static_assert(GRID0 * BLOCK == (KSH / 16) * (GN / 4), "retile tasks");

__device__ __forceinline__ void push_all(const Ctx& c, const Params& p, size_t bufoff, int cw) {
    const int lane = threadIdx.x & 63, wid = __builtin_amdgcn_readfirstlane(threadIdx.x >> 6);
    const int wv = cw * NWAVE + wid;
    char* peer[7];
#pragma unroll
    for (int i = 0; i < 7; ++i) { const unsigned pr = (unsigned)(c.rank + 1 + i) & 7u; peer[i] = (char*)p.ws_all[pr]; }
    char* mypeer = (char*)p.ws_all[(unsigned)(c.rank + 1 + (lane < 7 ? lane : 0)) & 7u];
    const size_t baseA = bufoff + (size_t)c.rank * A_SRC, baseB = bufoff + A_BYTES + (size_t)c.rank * B_SRC;
    LAS unsigned* arr = (LAS unsigned*)c.lds;
    if (threadIdx.x < NUNITS) arr[threadIdx.x] = 0u;
    __syncthreads();
    for (int u = 0; u < NUNITS; ++u) {
        int pm0 = 0, nA = 0, pn0, nB, kq0, nq = 2;
        if (u < 4) { pm0 = 0; nA = 8; pn0 = 0; nB = 8; kq0 = 2 * u; }
        else if (u < 9) { pm0 = 8; nA = 8; pn0 = 8; nB = 12; kq0 = u < 7 ? 2 * (u - 4) : u - 1; nq = u < 7 ? 2 : 1; }
        else { const int g = (u - 9) >> 2; pn0 = 20 + 16 * g; nB = g == 2 ? 12 : 16; kq0 = 2 * ((u - 9) & 3); }
        const int nP = nA + nB;
        const int ncell = nq * nP * 16;
#define WK_DRIFT 2
        if (u >= WK_DRIFT) while (__hip_atomic_load(arr + (u - WK_DRIFT), __ATOMIC_RELAXED, __HIP_MEMORY_SCOPE_WORKGROUP) < (unsigned)NWAVE) __builtin_amdgcn_s_sleep(1);
        size_t off[5]; u32x4 v[5];
#pragma unroll
        for (int i = 0; i < 5; ++i) {
            const int cell = wv + i * NPUSH, cl = cell < ncell ? cell : wv;
            const int pj = cl >> 4, ci = cl & 15, kq = kq0 + (pj >= nP ? 1 : 0), pi = pj >= nP ? pj - nP : pj;
            size_t o;
            if (pi < nA) o = baseA + (size_t)kq * KH_STR + (size_t)(pm0 + pi) * PANEL;
            else { const int pn = pn0 + pi - nA;
                   o = baseB + (size_t)(pn >> 4) * R_STR + (size_t)kq * KH_STR + (size_t)(pn & 15) * PANEL; }
            off[i] = o + (size_t)ci * 1024 + (size_t)lane * 16;
            if (cell < ncell) v[i] = *(const u32x4*)(c.ws + off[i]);
        }
#pragma unroll
        for (int i = 0; i < 5; ++i)
            if (wv + i * NPUSH < ncell) {
#pragma unroll
                for (int q = 0; q < 7; ++q)
                    asm volatile("global_store_dwordx4 %0, %1, off sc0 sc1\n\ts_nop 1" :: "v"(peer[q] + off[i]), "v"(v[i]) : "memory");
            }
        WAIT_V(0);
        unsigned prev = 0;
        if (lane == 0) prev = __hip_atomic_fetch_add(arr + u, 1u, __ATOMIC_RELAXED, __HIP_MEMORY_SCOPE_WORKGROUP);
        prev = __builtin_amdgcn_readfirstlane(prev);
        if (prev == (unsigned)(NWAVE - 1) && lane < 7)
            __hip_atomic_fetch_add((unsigned*)(mypeer + OFF_CNT) + u * 8 + c.rank, 1u, __ATOMIC_RELAXED, __HIP_MEMORY_SCOPE_SYSTEM);
    }
    __syncthreads();
}

__global__ void __launch_bounds__(BLOCK, WN / 2) fused_kernel(Params p, int phase) {
    extern __shared__ __attribute__((aligned(16))) unsigned char shm[];
    LAS unsigned char* lds = (LAS unsigned char*)shm;
    LAS unsigned* slot = (LAS unsigned*)(lds + LDS_STAGE + 16);
    if (phase == 0) {
        retile(p, slot);
        return;
    }
    Ctx c;
    c.call = call_number(p.ws, OFF_TICKET1, slot);
    const size_t bufoff = DATA_OFF + (size_t)(c.call & 1u) * BUF_BYTES;
    c.ws = p.ws; c.gA = p.ws + bufoff; c.gB = c.gA + A_BYTES; c.lds = lds;
    c.rank = p.rank; c.nranks = p.nranks; c.ready = -1; c.bad = false;
    const float scale = p.sx[0] * p.sw[0];
    const int xg = blockIdx.x & 7, j = blockIdx.x >> 3;
    STAMP(c, 0);
    if (j < CW) push_all(c, p, bufoff, j * 8 + xg);
    gemm_stream<4, 2>(c, p, scale, xg, -1 - (j >> 2), 0, 1, j & 3);
    STAMP(c, 30);
    if (j >= CW) gemm_stream<8, 1>(c, p, scale, xg, j - CW, GW, 4, 0);
    STAMP(c, 5);
}

}

extern "C" void kernel_launch(int rank, int nranks, ncclComm_t comm,
                              hipStream_t stream,
                              void* const* d_in, const int* in_sizes, int n_in,
                              void* d_out, int out_size,
                              const int* in_dtypes, int out_dtype,
                              void* const* d_out_all,
                              void* const* d_ws_all, size_t ws_bytes) {
    (void)comm; (void)in_sizes; (void)n_in; (void)out_size; (void)in_dtypes; (void)out_dtype; (void)d_out_all;
    if (nranks != wk::NR || ws_bytes < wk::WS_NEED || (unsigned)rank >= (unsigned)nranks) {
        fprintf(stderr, "unsupported configuration (nranks %d, ws %zu)\n", nranks, ws_bytes);
        return;
    }
    static bool attr_set = false;
    if (!attr_set)
    {
        HIP_CHECK(hipFuncSetAttribute((const void*)wk::fused_kernel, hipFuncAttributeMaxDynamicSharedMemorySize, (int)wk::LDS_BYTES));
        attr_set = true;
    }
    wk::Params p;
    p.x = (const int8_t*)d_in[0]; p.w = (const int8_t*)d_in[1];
    p.sx = (const float*)d_in[2]; p.sw = (const float*)d_in[3];
    p.out = (float*)d_out; p.ws = (char*)d_ws_all[rank]; p.ws_all = d_ws_all;
    p.rank = rank; p.nranks = nranks;
    hipLaunchKernelGGL(wk::fused_kernel, dim3(wk::GRID0), dim3(wk::BLOCK), wk::LDS_BYTES, stream, p, 0);
    hipLaunchKernelGGL(wk::fused_kernel, dim3(wk::GRID), dim3(wk::BLOCK), wk::LDS_BYTES, stream, p, 1);
    HIP_CHECK(hipGetLastError());
}
```
